```python
import numpy as np

kernel_name = "gpu_a2a_gemm_m4096_k4096_n4096_f32_silu"


ABSMAX_RTOL = 0.001

COMPUTE_DTYPE = "float32"
EPILOGUE = "silu"
IN_QUANT = None
OUT_QUANT = None
IN_DTYPES = {"x": "float32", "w_mat": "float32"}
WORLD_SIZE = 8

def setup_inputs():

    x = np.random.randn(4096, 4096).astype(np.float32)
    w_mat = (np.random.randn(4096, 4096) / np.sqrt(4096)).astype(np.float32)
    return {"x": x, "w_mat": w_mat}

def _epilogue(y):
    return (y / (1.0 + np.exp(-np.clip(y, -60.0, 60.0)))).astype(np.float32)

def reference(x, w_mat):
    y = (x @ w_mat).astype(np.float32)
    return _epilogue(y)
```

```cpp
#include <hip/hip_runtime.h>
#include <rccl/rccl.h>
#include <cstdio>
#include <type_traits>

#define HIP_CHECK(expr) do { hipError_t _e = (expr); if (_e != hipSuccess) { \
    fprintf(stderr, "HIP error %s at %s:%d\n", hipGetErrorString(_e), __FILE__, __LINE__); \
    return; } } while (0)

namespace {
typedef unsigned v4u __attribute__((ext_vector_type(4)));
typedef unsigned v2u __attribute__((ext_vector_type(2)));
typedef float f32x4 __attribute__((ext_vector_type(4)));
typedef float f32x2 __attribute__((ext_vector_type(2)));
typedef _Float16 h16x2 __attribute__((ext_vector_type(2)));
#define LAS __attribute__((address_space(3)))
#define GAS __attribute__((address_space(1)))

constexpr int NR = 8;
constexpr int MG = 4096, N = 4096, K = 4096;
constexpr int ML = MG / NR;
constexpr int KL = K / NR;
constexpr int NWG = 256, NTHR = 512;
constexpr int T = 32;
constexpr int FR = 4;
constexpr int PA = 4;
constexpr int PB = 4;
constexpr int RING = 19;
constexpr int ROT = 3;
constexpr int URG = 2;
constexpr int PACE = 40;
constexpr int LEAD_URG = 100, LEAD_REST = 250;
constexpr int PW = 8;
constexpr int PAHEAD = 12;
constexpr unsigned POLL0 = 250;
constexpr int RSRC_W3 = 0x00020000;
constexpr int AUX_SYS = 17;
constexpr int AUX_AG = 16;
constexpr int LDS_XT = RING * 8192;
constexpr int LDS_CTRL = LDS_XT + 4096;
constexpr int LDS_BYTES = LDS_CTRL + 256;
static_assert(2 * RING >= T && LDS_BYTES <= 163840 && T % PA == 0 && T % PB == 0 && PB == 4 && PAHEAD > PA + 1 && RING > PA, "config");
constexpr size_t WS_TMO = 0;
constexpr size_t WS_CNT = 4096;
constexpr size_t WS_XFLAG = 131072;
constexpr size_t WS_STAGE = 1u << 20;
constexpr size_t STAGE_BYTES = (size_t)ML * K * 2;
constexpr size_t WS_XCH = WS_STAGE + 3 * STAGE_BYTES;
constexpr size_t XCH_BYTES = (size_t)NWG * 4 * 3 * 8192;
constexpr size_t WS_NEED = WS_XCH + XCH_BYTES;
constexpr unsigned long long DEADLINE_TICKS = 50000000ull;

struct Params {
  const float* x;
  const float* w;
  float* out;
  unsigned char* ws[NR];
  int rank;
};

__device__ __forceinline__ unsigned cvt2(float a, float b) {
  f32x2 v = {a, b};
  return __builtin_bit_cast(unsigned, __builtin_convertvector(v, h16x2));
}
__device__ __forceinline__ unsigned cvt2nz(float a, float b) {
  const unsigned u = cvt2(a, b);
  return u == 0u ? 0x8000u : u;
}
__device__ __forceinline__ unsigned min4(unsigned m, v4u f) { return min(min(m, min(f[0], f[1])), min(f[2], f[3])); }
#define MFMA(c, a, b) asm volatile("v_mfma_f32_16x16x32_f16 %0, %1, %2, %0" : "+v"(c) : "v"(a), "v"(b))
__device__ __forceinline__ float silu(float y) {
  const float t = fminf(fmaxf(y, -60.f), 60.f);
  return y * __builtin_amdgcn_rcpf(1.0f + __expf(-t));
}
__device__ __forceinline__ int lds_ld(LAS int* p) {
  return __builtin_amdgcn_readfirstlane(__hip_atomic_load(p, __ATOMIC_RELAXED, __HIP_MEMORY_SCOPE_WORKGROUP));
}
__device__ __forceinline__ void lds_st(LAS int* p, int v) { __hip_atomic_store(p, v, __ATOMIC_RELAXED, __HIP_MEMORY_SCOPE_WORKGROUP); }
__device__ __forceinline__ int slot_of(int t) { return t < RING ? t : t - RING; }
#define CBAR() asm volatile("" ::: "memory")
#define SB() __builtin_amdgcn_sched_barrier(0)
#define WAITV(n) asm volatile("s_waitcnt vmcnt(" #n ")" ::: "memory")

enum { C_FULL = 0, C_USED = 4, C_AREADY = 8, C_CALL = 9, C_ABORT = 10, C_XISS = 11 };

__global__ void __launch_bounds__(NTHR, 2) a2a_gemm_silu(Params p) {
  extern __shared__ __attribute__((aligned(16))) unsigned char smem[];
  const unsigned long long tstart = __builtin_amdgcn_s_memrealtime();
  LAS unsigned char* L = (LAS unsigned char*)smem;
  LAS int* ctl = (LAS int*)(L + LDS_CTRL);
  const int tid = threadIdx.x, lane = tid & 63;
  const int w = __builtin_amdgcn_readfirstlane(tid >> 6);
  const int bid = blockIdx.x & (NWG - 1), xcd = bid & 7, idx = bid >> 3;
  const int kq = xcd >> 1, ntl = idx >> 1, mt = idx & 1, nt = (xcd & 1) * 16 + ntl;
  const int n0 = nt * 128;
  const int g = lane >> 4, c = lane & 15;
  const int rank = p.rank & (NR - 1);
  const int dd = bid >> 5;
  unsigned char* myws = p.ws[0];
  unsigned char* dws = p.ws[dd];
  GAS unsigned* tmo = (GAS unsigned*)(myws + WS_TMO);
  auto ks_of = [&](int t) {
    const int i = (t & 31) * 4 + kq;
    if (i < 16) return rank * 16 + i;
    const int u = i - 16, s = u / 7, v = u - s * 7;
    return ((rank + 7 - v) & 7) * 16 + ((s + ROT * (v + 1)) & 15);
  };
  auto expired = [&](unsigned long long t0, unsigned call) -> bool {
    if (__builtin_amdgcn_readfirstlane(__hip_atomic_load(tmo, __ATOMIC_RELAXED, __HIP_MEMORY_SCOPE_AGENT)) == call ||
        __builtin_amdgcn_s_memrealtime() - t0 > DEADLINE_TICKS) {
      if (lane == 0) { __hip_atomic_store(tmo, call, __ATOMIC_RELAXED, __HIP_MEMORY_SCOPE_AGENT); lds_st(ctl + C_ABORT, 1); }
      return true;
    }
    return false;
  };

  asm volatile("" :: "s"(p.x), "s"(p.w), "s"(p.out), "s"(rank), "s"(myws), "s"(dws));
  GAS unsigned* cnt = (GAS unsigned*)(myws + WS_CNT + (size_t)bid * 128);
  unsigned call0 = 0;
  if (w == 7) call0 = __hip_atomic_load(cnt, __ATOMIC_RELAXED, __HIP_MEMORY_SCOPE_AGENT);
  if (w < 4 || w == 7) {
    unsigned long long c0 = (unsigned long long)p.w + (size_t)idx * (2u << 20);
    unsigned long long c1 = (unsigned long long)myws + WS_STAGE + (size_t)(idx & 7) * (3u << 19);
    unsigned long long c2 = (unsigned long long)dws + WS_STAGE + (size_t)min(idx & 3, 2) * STAGE_BYTES + (size_t)rank * (1u << 19);
    unsigned long long c3 = (unsigned long long)myws + WS_XCH + (size_t)(idx & 15) * (3u << 19);
    unsigned long long a0 = (unsigned long long)p.out + (size_t)(idx & 3) * (2u << 20);
    asm volatile("" : "+s"(c0), "+s"(c1), "+s"(c2), "+s"(c3), "+s"(a0));
    a0 = w == 0 ? c0 : a0; asm volatile("" : "+s"(a0));
    a0 = w == 1 ? c1 : a0; asm volatile("" : "+s"(a0));
    a0 = w == 2 ? c2 : a0; asm volatile("" : "+s"(a0));
    a0 = w == 3 ? c3 : a0;
    unsigned d0;
    asm volatile("s_load_dword %0, %1, 0x0\n\ts_barrier\n\ts_waitcnt lgkmcnt(0)" : "=&s"(d0) : "s"(a0) : "memory");
  } else {
    if (tid >= 256 && tid < 272) ctl[tid - 256] = 0;
    asm volatile("s_waitcnt lgkmcnt(0)\n\ts_barrier" ::: "memory");
  }

  if (__builtin_expect(w < 4, 0)) {
    const int tab = ks_of(lane);
    f32x4 acc[FR][8];
#pragma unroll
    for (int i = 0; i < FR; ++i)
#pragma unroll
      for (int j = 0; j < 8; ++j) acc[i][j] = f32x4{0.f, 0.f, 0.f, 0.f};
    v4u a[PA][FR];
    int aready = 0, bavail = 0;
    auto need_a = [&](int t) {
      if (t >= aready) {
        do { aready = lds_ld(ctl + C_AREADY); if (t < aready) break; __builtin_amdgcn_s_sleep(1); } while (true);
      }
      CBAR();
    };
    auto need_b = [&](int t) {
      if (t >= bavail) {
        do {
          bavail = min(lds_ld(ctl + C_FULL), lds_ld(ctl + C_FULL + 1));
          if (t < bavail) break;
          __builtin_amdgcn_s_sleep(1);
        } while (true);
      }
      CBAR();
    };
    const int tabA = tab * 32768;
    const int tabS = slot_of(lane & 31) * 8192;
    need_a(0);
    const unsigned call = (unsigned)lds_ld(ctl + C_CALL);
    const int rb0 = mt * 16 + w * FR;
    const __amdgpu_buffer_rsrc_t rsA = __builtin_amdgcn_make_buffer_rsrc(
        (void*)(myws + WS_STAGE + (size_t)(call % 3u) * STAGE_BYTES + (size_t)rb0 * 1024), 0, (int)STAGE_BYTES - rb0 * 1024, RSRC_W3);
    int voA[FR];
#pragma unroll
    for (int f = 0; f < FR; ++f) voA[f] = lane * 16 + ((f + kq) & 3) * 1024;
    auto ld_a = [&](int so, int f) { return __builtin_amdgcn_raw_buffer_load_b128(rsA, voA[f], so, AUX_SYS); };
    auto repair = [&](int t, v4u (&f)[FR]) {
      const unsigned long long t0 = __builtin_amdgcn_s_memrealtime();
      const int so = __builtin_amdgcn_readlane(tabA, t);
      do {
        unsigned m = ~0u;
#pragma unroll
        for (int i = 0; i < FR; ++i) f[i] = ld_a(so, i);
#pragma unroll
        for (int i = 0; i < FR; ++i) m = min4(m, f[i]);
        if (__builtin_amdgcn_ballot_w64(m == 0u) == 0ull) break;
        if (expired(t0, call)) break;
      } while (true);
    };
#pragma unroll
    for (int i = 0; i < PA - 1; ++i) {
      need_a(i);
      const int so = __builtin_amdgcn_readlane(tabA, i);
#pragma unroll
      for (int f = 0; f < FR; ++f) a[i][f] = ld_a(so, f);
      SB();
    }
    const int ldsl = lane * 16;
    LAS int* usedp = ctl + C_USED + w;
    v4u bh[4];
    need_b(0);
#pragma unroll
    for (int j = 0; j < 3; ++j) bh[j] = *(LAS v4u*)(L + ldsl + j * 1024);
    {
      unsigned m = ~0u;
#pragma unroll
      for (int f = 0; f < FR; ++f) m = min4(m, a[0][f]);
      if (__builtin_expect(__builtin_amdgcn_ballot_w64(m == 0u) != 0ull, 0)) repair(0, a[0]);
    }
    LAS v4u* cur = (LAS v4u*)(L + ldsl);
    need_a(PA - 1);
    int so = __builtin_amdgcn_readlane(tabA, PA - 1);
#pragma unroll
    for (int i = 0; i < FR; ++i)
#pragma unroll
      for (int j = 0; j < 8; ++j) asm volatile("" : "+v"(acc[i][j]));
    asm volatile("s_nop 7" ::: "memory");
    auto body = [&](const int t0_, auto LAST) {
#pragma unroll
      for (int i = 0; i < PA; ++i) {
        const int t = t0_ + i;
        const bool nxt = !(LAST.value && i == PA - 1);
        const bool lda = !(LAST.value && i > 0);
        const bool ldn = !LAST.value;
        LAS v4u* nx = nullptr;
        unsigned m = ~0u;
#pragma unroll
        for (int j = 0; j < 8; ++j) {
#pragma unroll
          for (int f = 0; f < FR; ++f) {
            MFMA(acc[f][j], a[i][f], bh[j & 3]);
            SB();
            if (f == 0) { if (j + 3 < 8) bh[(j + 3) & 3] = cur[(j + 3) * 64]; else if (nxt) bh[(j + 3) & 3] = nx[(j + 3 - 8) * 64]; }
            if (f == 2 && j < 4 && lda) a[(i + PA - 1) % PA][j] = ld_a(so, j);
            if (f == 1 && j == 4 && nxt) need_b(t + 1);
            if (f == 2 && j == 4 && nxt) nx = (LAS v4u*)(L + __builtin_amdgcn_readlane(tabS, t + 1) + ldsl);
            if (f == 2 && j == 5 && !LAST.value) { CBAR(); lds_st(usedp, t + 1); CBAR(); }
            if (f == 2 && j == 6 && ldn) need_a(t + PA);
            if (f == 2 && j == 7 && ldn) so = __builtin_amdgcn_readlane(tabA, t + PA);
            if (f == 1 && j >= 5 && nxt) m = min4(m, a[(i + 1) % PA][j - 5]);
            if (f == 3 && j == 6 && nxt) m = min4(m, a[(i + 1) % PA][3]);
            SB();
          }
        }
        if (nxt) { if (__builtin_expect(__builtin_amdgcn_ballot_w64(m == 0u) != 0ull, 0)) repair(t + 1, a[(i + 1) % PA]); }
        cur = nx;
      }
    };
    for (int t0_ = 0; t0_ < T - PA; t0_ += PA) body(t0_, std::false_type{});
    body(T - PA, std::true_type{});
    asm volatile("s_nop 15\n\ts_nop 15" ::: "memory");
    {
      const __amdgpu_buffer_rsrc_t rsX = __builtin_amdgcn_make_buffer_rsrc((void*)(myws + WS_XCH), 0, (int)XCH_BYTES, RSRC_W3);
      GAS unsigned* fl = (GAS unsigned*)(myws + WS_XFLAG);
      int pb[4];
#pragma unroll
      for (int s = 1; s < 4; ++s) pb[s] = (bid & ~6) | (((kq + s) & 3) << 1);
#pragma unroll
      for (int s = 1; s < 4; ++s) {
        const int sx = ((pb[s] * 4 + w) * 3 + (s - 1)) * 8192;
#pragma unroll
        for (int j = 0; j < 8; ++j) __builtin_amdgcn_raw_buffer_store_b128(__builtin_bit_cast(v4u, acc[s][j]), rsX, lane * 16 + j * 1024, sx, AUX_AG);
        SB();
      }
      WAITV(16); if (lane == 0) __hip_atomic_store(fl + (pb[1] * 4 + w) * 32 + 0, call, __ATOMIC_RELAXED, __HIP_MEMORY_SCOPE_AGENT);
      WAITV(9);  if (lane == 0) __hip_atomic_store(fl + (pb[2] * 4 + w) * 32 + 1, call, __ATOMIC_RELAXED, __HIP_MEMORY_SCOPE_AGENT);
      WAITV(2);  if (lane == 0) __hip_atomic_store(fl + (pb[3] * 4 + w) * 32 + 2, call, __ATOMIC_RELAXED, __HIP_MEMORY_SCOPE_AGENT);
      const unsigned long long t0 = __builtin_amdgcn_s_memrealtime();
      unsigned spins = 0;
      while (true) {
        unsigned v = call;
        if (lane < 3) v = __hip_atomic_load(fl + (bid * 4 + w) * 32 + lane, __ATOMIC_RELAXED, __HIP_MEMORY_SCOPE_AGENT);
        if (__builtin_amdgcn_ballot_w64((int)(v - call) < 0) == 0ull) break;
        if ((++spins & 63) == 0 && expired(t0, call)) break;
      }
      CBAR();
      const int si = (bid * 4 + w) * 3 * 8192;
#pragma unroll
      for (int s = 1; s < 4; ++s)
#pragma unroll
        for (int j = 0; j < 8; ++j) acc[s][j] = __builtin_bit_cast(f32x4, __builtin_amdgcn_raw_buffer_load_b128(rsX, lane * 16 + j * 1024, si + (s - 1) * 8192, AUX_AG));
#pragma unroll
      for (int j = 0; j < 8; ++j) acc[0][j] += (acc[1][j] + acc[2][j]) + acc[3][j];
    }
    const bool bad = lds_ld(ctl + C_ABORT) != 0;
    const __amdgpu_buffer_rsrc_t rsC = __builtin_amdgcn_make_buffer_rsrc((void*)p.out, 0, ML * N * 4, RSRC_W3);
    const int row0 = (rb0 + kq) * 16;
#pragma unroll
    for (int hh = 0; hh < 2; ++hh)
#pragma unroll
      for (int reg = 0; reg < 4; ++reg) {
        f32x4 o = {silu(acc[0][hh * 4 + 0][reg]), silu(acc[0][hh * 4 + 1][reg]), silu(acc[0][hh * 4 + 2][reg]), silu(acc[0][hh * 4 + 3][reg])};
        if (bad) o = f32x4{__builtin_nanf(""), __builtin_nanf(""), __builtin_nanf(""), __builtin_nanf("")};
        const int row = row0 + g * 4 + reg;
        __builtin_amdgcn_raw_buffer_store_b128(__builtin_bit_cast(v4u, o), rsC, (row * N + n0 + hh * 64 + c * 4) * 4, 0, 0);
      }
    return;
  }
  if (__builtin_expect(w == 7, 0)) {
    const int tab = ks_of(lane);
    __builtin_amdgcn_s_setprio(3);
    const unsigned call = __builtin_amdgcn_readfirstlane(call0) + 1u;
    lds_st(ctl + C_CALL, (int)call);
    const unsigned char* base = myws + WS_STAGE + (size_t)(call % 3u) * STAGE_BYTES + mt * 16384;
    const int tabA = tab * 32768;
    int done = 0;
    unsigned att = 0;
    while (__builtin_amdgcn_s_memrealtime() - tstart < POLL0) __builtin_amdgcn_s_sleep(2);
    while (done < T) {
      if (done < lds_ld(ctl + C_USED) + PAHEAD) {
        const int lo = 16384 - 64 - (int)(((att + ntl * 8) & 127) * 64);
        unsigned v[PW];
#pragma unroll
        for (int i = 0; i < PW; ++i) {
          const unsigned char* ad = base + __builtin_amdgcn_readlane(tabA, min(done + i, T - 1)) + lo;
          asm volatile("s_load_dword %0, %1, 0x0" : "=s"(v[i]) : "s"(ad) : "memory");
        }
        asm volatile("s_waitcnt lgkmcnt(0)"
                     : "+s"(v[0]), "+s"(v[1]), "+s"(v[2]), "+s"(v[3]), "+s"(v[4]), "+s"(v[5]), "+s"(v[6]), "+s"(v[7]) :: "memory");
        int nd = done; bool go = true;
#pragma unroll
        for (int i = 0; i < PW; ++i) { go = go && v[i] != 0u; nd += go ? 1 : 0; }
        nd = min(nd, T);
        ++att;
        if ((att & 31) == 0 && nd == done) {
          const unsigned x = __hip_atomic_load((GAS unsigned*)(base + __builtin_amdgcn_readlane(tabA, done) + 16380), __ATOMIC_RELAXED, __HIP_MEMORY_SCOPE_SYSTEM);
          if (__builtin_amdgcn_readfirstlane(x) != 0u) nd = done + 1;
          else if ((att & 255) == 0 && expired(tstart, call)) { CBAR(); lds_st(ctl + C_AREADY, T); break; }
        }
        if (nd != done) {
          CBAR(); lds_st(ctl + C_AREADY, nd);
          done = nd;
          continue;
        }
      }
      __builtin_amdgcn_s_sleep(4);
    }
    if (lane == 0) __hip_atomic_store(cnt, call, __ATOMIC_RELAXED, __HIP_MEMORY_SCOPE_AGENT);
    return;
  }
  static_assert(PW == 8, "the wait statement above names 8 registers");

  const int sl = (bid >> 1) & 15, mh = bid & 1, kk = (sl + ROT * dd) & 15;
  const bool urgent = dd == 0 || sl < URG;
  f32x4 r[4][8];
  const bool isx = w == 6;
  const __amdgpu_buffer_rsrc_t rsL = __builtin_amdgcn_make_buffer_rsrc(
      isx ? (void*)(p.x + (size_t)((((rank + dd) & 7) * ML + mh * 256) * KL)) : (void*)(p.w + n0), 0, isx ? 256 * KL * 4 : K * N * 4 - n0 * 4, RSRC_W3);
  const int voL = isx ? (lane >> 3) * (KL * 4) + kk * 128 + (lane & 7) * 16 : ((w & 1) * 2 + (lane >> 5)) * 8 * N * 4 + (lane & 31) * 16;
  {
    if (isx) __builtin_amdgcn_s_setprio(3);
    if (isx != urgent) { while (lds_ld(ctl + C_XISS) < (urgent ? 2 : 1)) __builtin_amdgcn_s_sleep(1); CBAR(); }
    const int so0 = isx ? 0 : (rank * 16 + kq) * (32 * N * 4), sost = isx ? 131072 : 4 * (32 * N * 4);
#pragma unroll
    for (int i = 0; i < 4; ++i)
#pragma unroll
      for (int j = 0; j < 8; ++j) r[i][j] = __builtin_bit_cast(f32x4, __builtin_amdgcn_raw_buffer_load_b128(rsL, voL, so0 + i * sost + j * 16384, 0));
    SB();
    CBAR(); __hip_atomic_fetch_max(ctl + C_XISS, 1, __ATOMIC_RELAXED, __HIP_MEMORY_SCOPE_WORKGROUP);
  }
  if (w != 6) {
    const int q = w - 4;
    const int tabB = ks_of(lane) * (32 * N * 4);
    int freeupto = RING;
    auto body = [&](const int u0, auto LAST) {
#pragma unroll
      for (int i = 0; i < PB; ++i) {
        const int t = u0 + i;
        if (t >= freeupto) {
          do {
            const int m = min(min(lds_ld(ctl + C_USED), lds_ld(ctl + C_USED + 1)), min(lds_ld(ctl + C_USED + 2), lds_ld(ctl + C_USED + 3)));
            freeupto = m + RING;
            if (t < freeupto) break;
            __builtin_amdgcn_s_sleep(2);
          } while (true);
        }
        CBAR();
        LAS v4u* dst = (LAS v4u*)(L + slot_of(t) * 8192 + ((lane >> 4) & 1) * 4096 + ((q * 2 + (lane >> 5)) * 16 + c) * 16);
#pragma unroll
        for (int j = 0; j < 4; ++j) {
          v4u h;
#pragma unroll
          for (int jj = 0; jj < 4; ++jj) h[jj] = cvt2(r[i][2 * jj][j], r[i][2 * jj + 1][j]);
          dst[j * 64] = h;
        }
        CBAR();
        lds_st(ctl + C_FULL + q, t + 1);
        CBAR();
        SB();
        if (!LAST.value) {
          const int so = __builtin_amdgcn_readlane(tabB, t + PB);
#pragma unroll
          for (int j = 0; j < 8; ++j) r[i][j] = __builtin_bit_cast(f32x4, __builtin_amdgcn_raw_buffer_load_b128(rsL, voL, so + j * (N * 4), 0));
        }
        SB();
      }
    };
    for (int u0 = 0; u0 < T - PB; u0 += PB) body(u0, std::false_type{});
    body(T - PB, std::true_type{});
  }
  int w2 = w;
  asm volatile("" : "+s"(w2));
  if (w2 == 6) {
    v4u h[16];
    {
      LAS unsigned char* xt = L + LDS_XT;
      const int wo = (((lane & 7) >> 1) * 16 + (lane >> 3)) * 16 + (lane & 1) * 8;
#pragma unroll
      for (int i0 = 0; i0 < 16; i0 += 4) {
#pragma unroll
        for (int i = 0; i < 4; ++i)
#pragma unroll
          for (int hh = 0; hh < 2; ++hh) {
            const f32x4 v = r[(i0 + i) >> 2][((i0 + i) & 3) * 2 + hh];
            *(LAS v2u*)(xt + i * 1024 + hh * 128 + wo) = v2u{cvt2nz(v[0], v[1]), cvt2nz(v[2], v[3])};
          }
        CBAR();
#pragma unroll
        for (int i = 0; i < 4; ++i) h[i0 + i] = *(LAS v4u*)(xt + i * 1024 + lane * 16);
        CBAR();
      }
    }
    SB();
    int callv;
    do { callv = lds_ld(ctl + C_CALL); if (callv) break; __builtin_amdgcn_s_sleep(1); } while (true);
    const unsigned call = (unsigned)callv;
    const __amdgpu_buffer_rsrc_t rsD = __builtin_amdgcn_make_buffer_rsrc(
        (void*)(dws + WS_STAGE + (size_t)(call % 3u) * STAGE_BYTES + (size_t)((rank * 16 + kk) * 32 + mh * 16) * 1024), 0, 16384, RSRC_W3);
    asm volatile("s_waitcnt lgkmcnt(0)" ::: "memory");
    const unsigned long long tb = __builtin_amdgcn_s_memrealtime();
    const int delay = dd ? (sl * 2 + mh) * PACE - (urgent ? LEAD_URG : LEAD_REST) : 0;
    while ((long long)(__builtin_amdgcn_s_memrealtime() - tb) < delay) __builtin_amdgcn_s_sleep(1);
#pragma unroll
    for (int i = 0; i < 16; ++i) __builtin_amdgcn_raw_buffer_store_b128(h[i], rsD, lane * 16, i * 1024, AUX_SYS);
    CBAR(); lds_st(ctl + C_XISS, 2);
    const __amdgpu_buffer_rsrc_t rsZ = __builtin_amdgcn_make_buffer_rsrc(
        (void*)(myws + WS_STAGE + (size_t)((call + 2u) % 3u) * STAGE_BYTES + (size_t)bid * 16384), 0, 16384, RSRC_W3);
#pragma unroll
    for (int i = 0; i < 16; ++i) __builtin_amdgcn_raw_buffer_store_b128(v4u{0u, 0u, 0u, 0u}, rsZ, lane * 16, i * 1024, 0);
  }
}
}

extern "C" void kernel_launch(int rank, int nranks, ncclComm_t comm,
                              hipStream_t stream,
                              void* const* d_in, const int* in_sizes, int n_in,
                              void* d_out, int out_size,
                              const int* in_dtypes, int out_dtype,
                              void* const* d_out_all,
                              void* const* d_ws_all, size_t ws_bytes) {
  if (nranks != NR || (unsigned)rank >= (unsigned)NR || ws_bytes < WS_NEED || n_in < 2 ||
      in_sizes[0] != MG * KL || in_sizes[1] != K * N || out_size != ML * N) {
    fprintf(stderr, "kernel_launch: unexpected shapes\n");
    return;
  }
  HIP_CHECK(hipFuncSetAttribute((const void*)a2a_gemm_silu, hipFuncAttributeMaxDynamicSharedMemorySize, LDS_BYTES));
  Params p;
  p.x = (const float*)d_in[0];
  p.w = (const float*)d_in[1];
  p.out = (float*)d_out;
  for (int i = 0; i < NR; ++i) p.ws[i] = (unsigned char*)d_ws_all[(rank + i) & 7];
  p.rank = rank;
  a2a_gemm_silu<<<NWG, NTHR, LDS_BYTES, stream>>>(p);
  HIP_CHECK(hipGetLastError());
}
```
